```python
import numpy as np

kernel_name = "gpu_ag_gemm_m4096_k4096_n16384_e5m2e5m2_dqrelu"


ABSMAX_RTOL = 0.05

COMPUTE_DTYPE = "float32"
EPILOGUE = "dequant_relu"
IN_QUANT = None
OUT_QUANT = None
IN_FP8 = "e5m2e5m2"
IN_DTYPES = {"x": "e5m2", "w_mat": "e5m2",
             "scale_x": "float32", "scale_w": "float32"}
WORLD_SIZE = 8

def _snap_fp8(x, mant_bits, max_abs, min_normal):
    a = np.abs(x.astype(np.float64))
    _m, _e = np.frexp(a)
    q = np.where(a >= min_normal, np.ldexp(1.0, _e - 1 - mant_bits),
                 min_normal * 2.0 ** -mant_bits)
    snapped = np.minimum(np.round(a / q) * q, max_abs)
    return (np.sign(x) * snapped).astype(np.float32)

def setup_inputs():

    x = _snap_fp8(np.random.randn(4096, 4096).astype(np.float32), 2, 57344.0, 2.0 ** -14)
    w_mat = _snap_fp8(np.random.randn(4096, 16384).astype(np.float32), 2, 57344.0, 2.0 ** -14)
    scale_x = (0.02 * (1.0 + np.random.rand(1))).astype(np.float32)
    scale_w = (0.02 * (1.0 + np.random.rand(1))).astype(np.float32)
    return {"x": x, "w_mat": w_mat, "scale_x": scale_x, "scale_w": scale_w}

def reference(x, w_mat, scale_x, scale_w):
    acc = (x @ w_mat).astype(np.float32)
    y = (acc * (scale_x * scale_w)).astype(np.float32)
    return np.maximum(y, 0.0).astype(np.float32)
```

```cpp
#include <hip/hip_runtime.h>
#include <rccl/rccl.h>
#include <cstdio>
#include <cstdint>

#define HIP_CHECK(expr) do { hipError_t _e = (expr); if (_e != hipSuccess) { \
    fprintf(stderr, "HIP error %s at %s:%d\n", hipGetErrorString(_e), __FILE__, __LINE__); \
    return; } } while (0)

namespace ag {

#define LAS __attribute__((address_space(3)))
#define GAS __attribute__((address_space(1)))
typedef int v8i __attribute__((ext_vector_type(8)));
typedef int v4i __attribute__((ext_vector_type(4)));
typedef int v2i __attribute__((ext_vector_type(2)));
typedef unsigned v4u __attribute__((ext_vector_type(4)));
typedef float v16f __attribute__((ext_vector_type(16)));
typedef float v4f __attribute__((ext_vector_type(4)));

constexpr int NR = 8;
constexpr int MSH = 512;
constexpr int KD = 4096;
constexpr int NSH = 2048;
constexpr int NSTG = KD / 128;
constexpr int NWG = 256;
constexpr int NTHR = 512;
constexpr unsigned WOFF = 16384;
constexpr unsigned SLOT = 49152;
constexpr int RING = 3;
constexpr unsigned LDS_BYTES = SLOT * RING;

constexpr size_t STAG_BYTES = (size_t)MSH * KD;
constexpr size_t OFF_FLAGS = 2 * STAG_BYTES;
constexpr size_t OFF_CTR = OFF_FLAGS + NR * 4 * 128;
constexpr size_t OFF_TMO = OFF_CTR + NWG * 4;
constexpr size_t OFF_STAMP = OFF_TMO + 256;
constexpr size_t WS_NEED = OFF_STAMP + NWG * 16 * 8;
constexpr unsigned long long DEADLINE_TICKS = 30000000ull;

struct Params {
    const uint8_t* x;
    const uint8_t* w;
    const float* sx;
    const float* sw;
    float* out;
    uint8_t* myws;
    uint8_t* ws[NR];
    int rank;
};

template <int N> __device__ __forceinline__ void wait_vm() { asm volatile("s_waitcnt vmcnt(%0)" ::"i"(N) : "memory"); }
__device__ __forceinline__ void bar() {
    asm volatile("" ::: "memory");
    __builtin_amdgcn_s_barrier();
    asm volatile("" ::: "memory");
}
__device__ __forceinline__ void store16_sys(uint8_t* p, v4u v) {
    asm volatile("global_store_dwordx4 %0, %1, off sc0 sc1\n\ts_nop 1" ::"v"(p), "v"(v) : "memory");
}
#define AG_STORE 0
__device__ __forceinline__ void store_out(float* p, v4f v) {
    *(v4f*)p = v;
}
__device__ __forceinline__ unsigned long long rt() { return __builtin_amdgcn_s_memrealtime(); }
template <int OFF> __device__ __forceinline__ v4i lds_rd128(unsigned a) {
    v4i r; asm volatile("ds_read_b128 %0, %1 offset:%2" : "=&v"(r) : "v"(a), "i"(OFF) : "memory"); return r;
}
template <int OFF> __device__ __forceinline__ v2i lds_rdtr8(unsigned a) {
    v2i r; asm volatile("ds_read_b64_tr_b8 %0, %1 offset:%2" : "=&v"(r) : "v"(a), "i"(OFF) : "memory"); return r;
}
template <int N> __device__ __forceinline__ void wait_lgkm() { asm volatile("s_waitcnt lgkmcnt(%0)" ::"i"(N) : "memory"); }
#define SCHED_FENCE() __builtin_amdgcn_sched_barrier(0)
__device__ __forceinline__ v8i cat8(v4i a, v4i b) { return __builtin_shufflevector(a, b, 0, 1, 2, 3, 4, 5, 6, 7); }
__device__ __forceinline__ v8i cat8(v2i a, v2i b, v2i c, v2i d) {
    return cat8(__builtin_shufflevector(a, b, 0, 1, 2, 3), __builtin_shufflevector(c, d, 0, 1, 2, 3));
}
#define MFMA(w, x, c) __builtin_amdgcn_mfma_scale_f32_32x32x64_f8f6f4((w), (x), (c), 1, 1, 0, 0, 0, 0)

#define STAMP(i) do { } while (0)
#define AG_SKEW 8

struct Frag {
    v4i x00, x01, x10, x11;
    v2i w00, w01, w02, w03, w10, w11, w12, w13;
};

__global__ __launch_bounds__(NTHR, 2) void ag_gemm_kernel(Params p) {
    extern __shared__ __attribute__((aligned(16))) uint8_t lds_raw[];
    LAS uint8_t* lds = (LAS uint8_t*)lds_raw;
    const int tid = threadIdx.x;
    const int lane = tid & 63;
    const int wid = __builtin_amdgcn_readfirstlane(tid >> 6);
    const int b = blockIdx.x;
    const int g = b & 7, idx = b >> 3;
    const int rank = p.rank;
    uint8_t* myws = p.myws;
    STAMP(0);

    unsigned* ctr = (unsigned*)(myws + OFF_CTR) + b;
    unsigned* tmo = (unsigned*)(myws + OFF_TMO);
    if (wid == 7) __builtin_amdgcn_fence(__ATOMIC_ACQUIRE, "");
    const float sx_v = p.sx[0], sw_v = p.sw[0];
    unsigned ctr_v;
    unsigned call;

    if (b < 128 && wid < 4) {
        const int crb = b >> 5, ckc = b & 31;
        const uint8_t* src = p.x + (size_t)crb * 128 * KD + ckc * 128;
        v4u d[4];
        const int cu0 = wid * 256 + lane;
#pragma unroll
        for (int i = 0; i < 4; ++i) {
            const int r = (cu0 >> 3) + 8 * i;
            d[i] = *(const v4u*)(src + (unsigned)(r * KD + (((lane & 7) ^ ((r >> 1) & 7)) << 4)));
        }
        asm volatile("global_load_dword %0, %1, off sc1" : "=v"(ctr_v) : "v"(ctr) : "memory");
        unsigned* f = (unsigned*)(p.ws[lane & 7] + OFF_FLAGS) + (rank * 4 + crb) * 32 + ckc;
        asm volatile("s_waitcnt vmcnt(0)" : "+v"(ctr_v), "+v"(d[0]), "+v"(d[1]), "+v"(d[2]), "+v"(d[3]), "+v"(f)::"memory");
        call = __builtin_amdgcn_readfirstlane(ctr_v) + 1u;
        uint8_t* dst = myws + (size_t)(call & 1u) * STAG_BYTES + (size_t)b * 16384 + cu0 * 16;
#pragma unroll
        for (int i = 0; i < 4; ++i) store16_sys(dst + 1024 * i, d[i]);
        wait_vm<0>();
        bar();
        if (wid == 0 && lane < NR && lane != rank) __hip_atomic_store(f, call, __ATOMIC_RELAXED, __HIP_MEMORY_SCOPE_SYSTEM);
    } else {
        asm volatile("global_load_dword %0, %1, off sc1" : "=v"(ctr_v) : "v"(ctr) : "memory");
        bar();
        asm volatile("s_waitcnt vmcnt(0)" : "+v"(ctr_v)::"memory");
        call = __builtin_amdgcn_readfirstlane(ctr_v) + 1u;
    }
    float scale = sx_v * sw_v;
    asm volatile("" : "+v"(scale));
    STAMP(1);

    const int u0 = wid * 128 + lane;
    const int r0 = u0 >> 3, r1 = r0 + 8, pos = lane & 7;
    const unsigned xsw0 = (unsigned)(r0 * KD + ((pos ^ ((r0 >> 1) & 7)) << 4));
    const unsigned xsw1 = (unsigned)(r1 * KD + ((pos ^ ((r1 >> 1) & 7)) << 4));

    const unsigned peer = (unsigned)(rank + g) & 7u;
    const int rb = idx >> 3, cb = idx & 7;
    const bool local = (g == 0);

    const uint8_t* aBase;
    unsigned aoff0, aoff1, aStride;
    if (local) {
        aBase = p.x + (size_t)rb * 128 * KD;
        aoff0 = xsw0; aoff1 = xsw1; aStride = 128u;
    } else {
        aBase = p.ws[peer] + (size_t)(call & 1u) * STAG_BYTES + (size_t)rb * 32 * 16384;
        aoff0 = (unsigned)u0 * 16u; aoff1 = aoff0 + 1024u; aStride = 16384u;
    }
    const int wr0 = wid * 16 + (lane >> 4);
    const unsigned boff0 = (unsigned)((wr0 + 0) * NSH + (((lane & 15) ^ (((wr0 + 0) & 7) << 1)) << 4));
    const unsigned boff1 = (unsigned)((wr0 + 4) * NSH + (((lane & 15) ^ (((wr0 + 4) & 7) << 1)) << 4));
    const uint8_t* bBase = p.w + cb * 256;
    const unsigned ldsx = (unsigned)wid * 2048u, ldsw = WOFF + (unsigned)wid * 4096u;

#define GLDS(gp, lo) __builtin_amdgcn_global_load_lds((const unsigned*)(gp), (LAS unsigned*)(lds + (lo)), 16, 0, 0)
#define ISSUE_A(S, SL) do { const uint8_t* _a = aBase + (size_t)(S) * aStride; \
        GLDS(_a + aoff0, (SL) * SLOT + ldsx); GLDS(_a + aoff1, (SL) * SLOT + ldsx + 1024); } while (0)
#define ISSUE_B01(S, SL) do { const uint8_t* _b = bBase + (size_t)(S) * (128 * NSH); \
        GLDS(_b + boff0, (SL) * SLOT + ldsw); GLDS(_b + boff1, (SL) * SLOT + ldsw + 1024); } while (0)
#define ISSUE_B23(S, SL) do { const uint8_t* _b = bBase + (size_t)(S) * (128 * NSH) + 8 * NSH; \
        GLDS(_b + boff0, (SL) * SLOT + ldsw + 2048); GLDS(_b + boff1, (SL) * SLOT + ldsw + 3072); } while (0)

    ISSUE_B01(0, 0); ISSUE_B23(0, 0); ISSUE_B01(1, 1); ISSUE_B23(1, 1); ISSUE_B01(2, 2); ISSUE_B23(2, 2);

    if (!local && wid == 0) {
        const unsigned* f = (lane < 32) ? (const unsigned*)(myws + OFF_FLAGS) + (peer * 4 + rb) * 32 + lane : (const unsigned*)tmo;
        const unsigned long long t0 = rt();
        for (unsigned it = 1;; ++it) {
            const unsigned v = __hip_atomic_load(f, __ATOMIC_RELAXED, __HIP_MEMORY_SCOPE_SYSTEM);
            if (__all(lane >= 32 || (int)(v - call) >= 0)) break;
            if (__any(lane >= 32 && v == call) || ((it & 15u) == 0u && rt() - t0 > DEADLINE_TICKS)) {
                if (lane == 0) __hip_atomic_store(tmo, call, __ATOMIC_RELAXED, __HIP_MEMORY_SCOPE_AGENT);
                break;
            }
            __builtin_amdgcn_s_sleep(1);
        }
        for (int i = 0; i < rb; ++i) __builtin_amdgcn_s_sleep(AG_SKEW);
    }
    wait_vm<0>();
    bar();
    STAMP(2);
    if (tid == 0) __hip_atomic_store(ctr, call, __ATOMIC_RELAXED, __HIP_MEMORY_SCOPE_AGENT);
    const bool bad = (unsigned)__builtin_amdgcn_readfirstlane(__hip_atomic_load(tmo, __ATOMIC_RELAXED, __HIP_MEMORY_SCOPE_AGENT)) == call;

    const int wm = wid >> 2, wn = wid & 3;
    const int h = lane >> 5;
    float* outp = p.out + ((size_t)peer * MSH + rb * 128 + wm * 64 + (lane & 31)) * NSH + cb * 256 + wn * 64 + 4 * h;

    if (bad) {
        wait_vm<0>();
        const float qn = __builtin_nanf("");
        const v4f nv = {qn, qn, qn, qn};
        for (int mb = 0; mb < 2; ++mb)
            for (int q = 0; q < 8; ++q) *(v4f*)(outp + (size_t)(mb * 32) * NSH + q * 8) = nv;
        return;
    }

    ISSUE_A(0, 0); ISSUE_A(1, 1); ISSUE_A(2, 2);

    const unsigned ldsBase = (unsigned)(uintptr_t)lds;
    const int sw = ((lane & 31) >> 1) & 7;
    const unsigned xrow = ldsBase + (unsigned)(wm * 64 + (lane & 31)) * 128u;
    const unsigned xa0 = xrow + (unsigned)(((2 * h + 0) ^ sw) << 4);
    const unsigned xa1 = xrow + (unsigned)(((2 * h + 1) ^ sw) << 4);
    const int g16 = lane >> 4, i16 = lane & 15, q8 = i16 >> 1;
    const unsigned wa0 = ldsBase + WOFF + (unsigned)(32 * h + q8) * 256u +
                         (unsigned)(((wn * 4 + (g16 & 1)) ^ (q8 << 1)) << 4) + (unsigned)(i16 & 1) * 8u;

    v16f acc00, acc01, acc10, acc11;
#pragma unroll
    for (int i = 0; i < 16; ++i) { acc00[i] = 0.f; acc01[i] = 0.f; acc10[i] = 0.f; acc11[i] = 0.f; }

#define RD_G0(F, KK, so) do { const unsigned _w = wa0 + (so), _x0 = (xa0 + (so)) ^ ((KK) * 64u), _x1 = (xa1 + (so)) ^ ((KK) * 64u); \
        F.w00 = lds_rdtr8<(KK) * 16384 + 0>(_w); F.w01 = lds_rdtr8<(KK) * 16384 + 2048>(_w); \
        F.w02 = lds_rdtr8<(KK) * 16384 + 4096>(_w); F.w03 = lds_rdtr8<(KK) * 16384 + 6144>(_w); \
        F.x00 = lds_rd128<0>(_x0); F.x01 = lds_rd128<0>(_x1); } while (0)
#define RD_G1(F, KK, so) do { const unsigned _x0 = (xa0 + (so)) ^ ((KK) * 64u), _x1 = (xa1 + (so)) ^ ((KK) * 64u); \
        F.x10 = lds_rd128<4096>(_x0); F.x11 = lds_rd128<4096>(_x1); } while (0)
#define RD_G2(F, KK, so) do { const unsigned _w = (wa0 + (so)) ^ 32u; \
        F.w10 = lds_rdtr8<(KK) * 16384 + 0>(_w); F.w11 = lds_rdtr8<(KK) * 16384 + 2048>(_w); \
        F.w12 = lds_rdtr8<(KK) * 16384 + 4096>(_w); F.w13 = lds_rdtr8<(KK) * 16384 + 6144>(_w); } while (0)
#define MM00(F) acc00 = MFMA(cat8(F.w00, F.w01, F.w02, F.w03), cat8(F.x00, F.x01), acc00)
#define MM10(F) acc10 = MFMA(cat8(F.w00, F.w01, F.w02, F.w03), cat8(F.x10, F.x11), acc10)
#define MM01(F) acc01 = MFMA(cat8(F.w10, F.w11, F.w12, F.w13), cat8(F.x00, F.x01), acc01)
#define MM11(F) acc11 = MFMA(cat8(F.w10, F.w11, F.w12, F.w13), cat8(F.x10, F.x11), acc11)

    Frag fa, fb;
    wait_vm<4>();
    bar();
    RD_G0(fa, 0, 0u); RD_G1(fa, 0, 0u); RD_G2(fa, 0, 0u);
    wait_lgkm<0>();
    SCHED_FENCE();

    unsigned so = 0;
    int sl = 0;
#define STAGE(S, NV, ISSUE) do { \
        const unsigned son = (so == 2 * SLOT) ? 0u : so + SLOT; \
        RD_G0(fb, 1, so); \
        SCHED_FENCE(); MM00(fa); SCHED_FENCE(); \
        RD_G1(fb, 1, so); RD_G2(fb, 1, so); \
        SCHED_FENCE(); MM10(fa); SCHED_FENCE(); \
        MM01(fa); SCHED_FENCE(); \
        MM11(fa); SCHED_FENCE(); \
        wait_vm<NV>(); \
        wait_lgkm<0>(); \
        bar(); \
        SCHED_FENCE(); \
        MM00(fb); SCHED_FENCE(); \
        RD_G0(fa, 0, son); \
        if (ISSUE) ISSUE_A((S) + 3, sl); \
        SCHED_FENCE(); MM10(fb); SCHED_FENCE(); \
        RD_G1(fa, 0, son); RD_G2(fa, 0, son); \
        if (ISSUE) ISSUE_B01((S) + 3, sl); \
        SCHED_FENCE(); MM01(fb); SCHED_FENCE(); \
        if (ISSUE) ISSUE_B23((S) + 3, sl); \
        SCHED_FENCE(); MM11(fb); SCHED_FENCE(); \
        wait_lgkm<0>(); \
        SCHED_FENCE(); \
        so = son; \
        sl = (sl == 2) ? 0 : sl + 1; } while (0)

    STAGE(0, 2, true);
    for (int S = 1; S < NSTG - 3; ++S) STAGE(S, 6, true);
    STAGE(NSTG - 3, 6, false);
    STAGE(NSTG - 2, 0, false);
    RD_G0(fb, 1, so); RD_G1(fb, 1, so); RD_G2(fb, 1, so);
    SCHED_FENCE();
    MM00(fa); MM10(fa); MM01(fa); MM11(fa);
    SCHED_FENCE();
    wait_lgkm<0>();
    SCHED_FENCE();
    MM00(fb); MM10(fb); MM01(fb); MM11(fb);
    STAMP(3);

#define EPI(ACC, MB, NB) do { float* _o = outp + (size_t)((MB) * 32) * NSH + (NB) * 32; \
        _Pragma("unroll") for (int q = 0; q < 4; ++q) { v4f v; \
            _Pragma("unroll") for (int j = 0; j < 4; ++j) { const float y = ACC[q * 4 + j] * scale; v[j] = y < 0.f ? 0.f : y; } \
            store_out(_o + q * 8, v); } } while (0)
    EPI(acc00, 0, 0); EPI(acc01, 0, 1); EPI(acc10, 1, 0); EPI(acc11, 1, 1);
}

}

extern "C" void kernel_launch(int rank, int nranks, ncclComm_t comm,
                              hipStream_t stream,
                              void* const* d_in, const int* in_sizes, int n_in,
                              void* d_out, int out_size,
                              const int* in_dtypes, int out_dtype,
                              void* const* d_out_all,
                              void* const* d_ws_all, size_t ws_bytes) {
    (void)comm; (void)in_dtypes; (void)out_dtype; (void)d_out_all;
    if (nranks != ag::NR || n_in < 4 || ws_bytes < ag::WS_NEED || (unsigned)rank >= (unsigned)ag::NR ||
        in_sizes[0] != ag::MSH * ag::KD || in_sizes[1] != ag::KD * ag::NSH || out_size != ag::NR * ag::MSH * ag::NSH) {
        fprintf(stderr, "kernel_launch: unexpected problem shape\n");
        return;
    }
    {
        static bool attr_set[64];
        int dev = 0;
        HIP_CHECK(hipGetDevice(&dev));
        if ((unsigned)dev >= 64u || !attr_set[dev]) {
            HIP_CHECK(hipFuncSetAttribute((const void*)ag::ag_gemm_kernel, hipFuncAttributeMaxDynamicSharedMemorySize, (int)ag::LDS_BYTES));
            if ((unsigned)dev < 64u) attr_set[dev] = true;
        }
    }
    ag::Params p;
    p.x = (const uint8_t*)d_in[0];
    p.w = (const uint8_t*)d_in[1];
    p.sx = (const float*)d_in[2];
    p.sw = (const float*)d_in[3];
    p.out = (float*)d_out;
    for (int i = 0; i < ag::NR; ++i) p.ws[i] = (uint8_t*)d_ws_all[i];
    p.myws = (uint8_t*)d_ws_all[rank];
    p.rank = rank;
    hipLaunchKernelGGL(ag::ag_gemm_kernel, dim3(ag::NWG), dim3(ag::NTHR), ag::LDS_BYTES, stream, p);
    HIP_CHECK(hipGetLastError());
}
```

```cpp
#include <hip/hip_runtime.h>
#include <rccl/rccl.h>
#include <cstdio>
#include <cstdint>

#define HIP_CHECK(expr) do { hipError_t _e = (expr); if (_e != hipSuccess) { \
    fprintf(stderr, "HIP error %s at %s:%d\n", hipGetErrorString(_e), __FILE__, __LINE__); \
    return; } } while (0)

namespace ag {

#define LAS __attribute__((address_space(3)))
typedef int v8i __attribute__((ext_vector_type(8)));
typedef int v4i __attribute__((ext_vector_type(4)));
typedef int v2i __attribute__((ext_vector_type(2)));
typedef unsigned v4u __attribute__((ext_vector_type(4)));
typedef float v16f __attribute__((ext_vector_type(16)));
typedef float v4f __attribute__((ext_vector_type(4)));

constexpr int NR = 8;
constexpr int MSH = 512;
constexpr int KD = 4096;
constexpr int NSH = 2048;
constexpr int NSTG = KD / 128;
constexpr int NWG = 256;
constexpr int NTHR = 512;
constexpr unsigned XSLOT = 16384;
constexpr unsigned WSLOT = 32768;
constexpr int XR = 4, WR = 3;
constexpr unsigned WOFF = XR * XSLOT;
constexpr unsigned LDS_BYTES = WOFF + WR * WSLOT;

constexpr int nv(int S) {
    int n = 0, last = 0;
    for (int i = 0; i < WR; ++i) { n += 4; if (i == S + 1) last = n; }
    for (int i = 0; i < XR; ++i) { n += 2; if (i == S + 1) last = n; }
    for (int s = 0; s < S; ++s) {
        if (s + WR < NSTG) { n += 4; if (s + WR == S + 1) last = n; }
        if (s + XR < NSTG) { n += 2; if (s + XR == S + 1) last = n; }
    }
    return n - last;
}
constexpr bool nv_const(int a, int b) { for (int s = a; s < b; ++s) if (nv(s) != nv(a)) return false; return true; }

constexpr size_t STAG_BYTES = (size_t)MSH * KD;
constexpr size_t OFF_FLAGS = 2 * STAG_BYTES;
constexpr size_t OFF_CTR = OFF_FLAGS + NR * 4 * 128;
constexpr size_t OFF_TMO = OFF_CTR + NWG * 4;
constexpr size_t OFF_STAMP = OFF_TMO + 256;
constexpr size_t WS_NEED = OFF_STAMP + NWG * 16 * 8;
constexpr unsigned long long DEADLINE_TICKS = 30000000ull;
#define PIECE(rb, kc) ((kc) * 4 + (rb))
#define AG_SKEW 0

struct Params {
    const uint8_t* x;
    const uint8_t* w;
    const float* sx;
    const float* sw;
    float* out;
    uint8_t* myws;
    uint8_t* ws[NR];
    int rank;
};

template <int N> __device__ __forceinline__ void wait_vm() { asm volatile("s_waitcnt vmcnt(%0)" ::"i"(N) : "memory"); }
template <int N> __device__ __forceinline__ void wait_lgkm() { asm volatile("s_waitcnt lgkmcnt(%0)" ::"i"(N) : "memory"); }
__device__ __forceinline__ void bar() {
    asm volatile("" ::: "memory");
    __builtin_amdgcn_s_barrier();
    asm volatile("" ::: "memory");
}
__device__ __forceinline__ void store16_sys(uint8_t* p, v4u v) {
    asm volatile("global_store_dwordx4 %0, %1, off sc0 sc1\n\ts_nop 1" ::"v"(p), "v"(v) : "memory");
}
__device__ __forceinline__ unsigned long long rt() { return __builtin_amdgcn_s_memrealtime(); }
template <int OFF> __device__ __forceinline__ v4i lds_rd128(unsigned a) {
    v4i r; asm volatile("ds_read_b128 %0, %1 offset:%2" : "=&v"(r) : "v"(a), "i"(OFF) : "memory"); return r;
}
template <int OFF> __device__ __forceinline__ v2i lds_rdtr8(unsigned a) {
    v2i r; asm volatile("ds_read_b64_tr_b8 %0, %1 offset:%2" : "=&v"(r) : "v"(a), "i"(OFF) : "memory"); return r;
}
__device__ __forceinline__ v8i cat8(v4i a, v4i b) { return __builtin_shufflevector(a, b, 0, 1, 2, 3, 4, 5, 6, 7); }
__device__ __forceinline__ v8i cat8(v2i a, v2i b, v2i c, v2i d) {
    return cat8(__builtin_shufflevector(a, b, 0, 1, 2, 3), __builtin_shufflevector(c, d, 0, 1, 2, 3));
}
__device__ __forceinline__ void mfma(v16f& c, v8i x, v8i w) {
    asm volatile("v_mfma_f32_32x32x64_f8f6f4 %0, %1, %2, %0 cbsz:1 blgp:1" : "+v"(c) : "v"(x), "v"(w));
}
#define SCHED_FENCE() __builtin_amdgcn_sched_barrier(0)

#define STAMP(i) do { } while (0)

struct Frag {
    v4i x00, x01, x10, x11;
    v2i w00, w01, w02, w03, w10, w11, w12, w13;
};

__global__ __launch_bounds__(NTHR, 2) void ag_gemm_kernel(Params p) {
    extern __shared__ __attribute__((aligned(16))) uint8_t lds_raw[];
    LAS uint8_t* lds = (LAS uint8_t*)lds_raw;
    const int tid = threadIdx.x;
    const int lane = tid & 63;
    const int wid = __builtin_amdgcn_readfirstlane(tid >> 6);
    const int b = blockIdx.x;
    const int g = b & 7, idx = b >> 3;
    const int rank = p.rank;
    uint8_t* myws = p.myws;
    STAMP(0);

    unsigned* ctr = (unsigned*)(myws + OFF_CTR) + b;
    unsigned* tmo = (unsigned*)(myws + OFF_TMO);
    const float sx_v = p.sx[0], sw_v = p.sw[0];
    unsigned ctr_v;
    unsigned call;

    if (b < 128 && wid < 4) {
        const int crb = b >> 5, ckc = b & 31;
        const uint8_t* src = p.x + (size_t)crb * 128 * KD + ckc * 128;
        v4u d[4];
        const int cu0 = wid * 256 + lane;
#pragma unroll
        for (int i = 0; i < 4; ++i) {
            const int r = (cu0 >> 3) + 8 * i;
            d[i] = *(const v4u*)(src + (unsigned)(r * KD + (((lane & 7) ^ ((r >> 1) & 7)) << 4)));
        }
        asm volatile("global_load_dword %0, %1, off sc1" : "=v"(ctr_v) : "v"(ctr) : "memory");
        unsigned* f = (unsigned*)(p.ws[lane & 7] + OFF_FLAGS) + (rank * 4 + crb) * 32 + ckc;
        asm volatile("s_waitcnt vmcnt(0)" : "+v"(ctr_v), "+v"(d[0]), "+v"(d[1]), "+v"(d[2]), "+v"(d[3]), "+v"(f)::"memory");
        call = __builtin_amdgcn_readfirstlane(ctr_v) + 1u;
        STAMP(6);
        uint8_t* dst = myws + (size_t)(call & 1u) * STAG_BYTES + (size_t)PIECE(crb, ckc) * 16384 + cu0 * 16;
#pragma unroll
        for (int i = 0; i < 4; ++i) store16_sys(dst + 1024 * i, d[i]);
        wait_vm<0>();
        bar();
        if (wid == 0 && lane < NR && lane != rank) __hip_atomic_store(f, call, __ATOMIC_RELAXED, __HIP_MEMORY_SCOPE_SYSTEM);
    } else
    {
        asm volatile("global_load_dword %0, %1, off sc1" : "=v"(ctr_v) : "v"(ctr) : "memory");
        if (b < 128) bar();
        asm volatile("s_waitcnt vmcnt(0)" : "+v"(ctr_v)::"memory");
        call = __builtin_amdgcn_readfirstlane(ctr_v) + 1u;
        STAMP(6);
    }
    float scale = sx_v * sw_v;
    asm volatile("" : "+v"(scale));
    STAMP(1);

    const unsigned peer = (unsigned)(rank + g) & 7u;
    const int rb = idx >> 3, cb = idx & 7;
    const bool local = (g == 0);

    const int u0 = wid * 128 + lane;
    const int r0 = u0 >> 3, r1 = r0 + 8, pos = lane & 7;
    const uint8_t* aBase;
    unsigned aoff0, aoff1, aStride;
    if (local) {
        aBase = p.x + (size_t)rb * 128 * KD;
        aoff0 = (unsigned)(r0 * KD + ((pos ^ ((r0 >> 1) & 7)) << 4));
        aoff1 = (unsigned)(r1 * KD + ((pos ^ ((r1 >> 1) & 7)) << 4));
        aStride = 128u;
    } else {
        aBase = p.ws[peer] + (size_t)(call & 1u) * STAG_BYTES + (size_t)PIECE(rb, 0) * 16384;
        aoff0 = (unsigned)u0 * 16u; aoff1 = aoff0 + 1024u;
        aStride = (PIECE(0, 1) - PIECE(0, 0)) * 16384u;
    }
    const int wr0 = wid * 16 + (lane >> 4);
    const unsigned boff0 = (unsigned)((wr0 + 0) * NSH + (((lane & 15) ^ (((wr0 + 0) & 7) << 1)) << 4));
    const unsigned boff1 = (unsigned)((wr0 + 4) * NSH + (((lane & 15) ^ (((wr0 + 4) & 7) << 1)) << 4));
    const uint8_t* bBase = p.w + cb * 256;
    const unsigned ldsx = (unsigned)wid * 2048u, ldsw = WOFF + (unsigned)wid * 4096u;

#define GLDS(gp, lo) __builtin_amdgcn_global_load_lds((const unsigned*)(gp), (LAS unsigned*)(lds + (lo)), 16, 0, 0)
#define ISSUE_X(S, xo) do { const uint8_t* _a = aBase + (size_t)(S) * aStride; \
        GLDS(_a + aoff0, (xo) + ldsx); GLDS(_a + aoff1, (xo) + ldsx + 1024); } while (0)
#define ISSUE_W01(S, wo) do { const uint8_t* _b = bBase + (size_t)(S) * (128 * NSH); \
        GLDS(_b + boff0, (wo) + ldsw); GLDS(_b + boff1, (wo) + ldsw + 1024); } while (0)
#define ISSUE_W23(S, wo) do { const uint8_t* _b = bBase + (size_t)(S) * (128 * NSH) + 8 * NSH; \
        GLDS(_b + boff0, (wo) + ldsw + 2048); GLDS(_b + boff1, (wo) + ldsw + 3072); } while (0)

#pragma unroll
    for (int s = 0; s < WR; ++s) { ISSUE_W01(s, s * WSLOT); ISSUE_W23(s, s * WSLOT); }

    if (!local && wid == 0) {
        const unsigned* fl = (lane < 32) ? (const unsigned*)(myws + OFF_FLAGS) + (peer * 4 + rb) * 32 + lane : (const unsigned*)tmo;
        const unsigned long long t0 = rt();
        for (unsigned it = 1;; ++it) {
            const unsigned v = __hip_atomic_load(fl, __ATOMIC_RELAXED, __HIP_MEMORY_SCOPE_SYSTEM);
            if (__any(lane >= 32 && v == call)) break;
            if (__all(lane >= 32 || (int)(v - call) >= 0)) break;
            if ((it & 15u) == 0u && rt() - t0 > DEADLINE_TICKS) {
                if (lane == 0) __hip_atomic_store(tmo, call, __ATOMIC_RELAXED, __HIP_MEMORY_SCOPE_AGENT);
                break;
            }
            __builtin_amdgcn_s_sleep(1);
        }
        for (int i = (AG_SKEW >> (4 * rb)) & 15; i > 0; --i) __builtin_amdgcn_s_sleep(8);
        wait_vm<0>();
    }
    bar();
    STAMP(2);
    if (tid == 0) __hip_atomic_store(ctr, call, __ATOMIC_RELAXED, __HIP_MEMORY_SCOPE_AGENT);
    unsigned tmo_v;
    asm volatile("global_load_dword %0, %1, off sc1" : "=v"(tmo_v) : "v"(tmo) : "memory");

#pragma unroll
    for (int s = 0; s < XR; ++s) ISSUE_X(s, s * XSLOT);

    const int wm = wid >> 2, wn = wid & 3;
    const int h = lane >> 5;
    const float qn = __builtin_nanf("");
#define OUT_PTR(L) (p.out + ((size_t)peer * MSH + rb * 128 + wm * 64 + 4 * ((L) >> 5)) * NSH + cb * 256 + wn * 64 + ((L) & 31))

    const unsigned ldsBase = (unsigned)(uintptr_t)lds;
    const int sw = ((lane & 31) >> 1) & 7;
    const unsigned xrow = ldsBase + (unsigned)(wm * 64 + (lane & 31)) * 128u;
    const unsigned xa0 = xrow + (unsigned)(((2 * h + 0) ^ sw) << 4);
    const unsigned xa1 = xrow + (unsigned)(((2 * h + 1) ^ sw) << 4);
    const int g16 = lane >> 4, i16 = lane & 15, q8 = i16 >> 1;
    const unsigned wa0 = ldsBase + WOFF + (unsigned)(32 * h + q8) * 256u +
                         (unsigned)(((wn * 4 + (g16 & 1)) ^ (q8 << 1)) << 4) + (unsigned)(i16 & 1) * 8u;

    v16f acc00, acc01, acc10, acc11;
#pragma unroll
    for (int i = 0; i < 16; ++i) { acc00[i] = 0.f; acc01[i] = 0.f; acc10[i] = 0.f; acc11[i] = 0.f; }
    asm volatile("s_nop 1" : "+v"(acc00), "+v"(acc01), "+v"(acc10), "+v"(acc11));

#define RD_G0(F, KK, xo, wo) do { const unsigned _w = wa0 + (wo), _x0 = (xa0 + (xo)) ^ ((KK) * 64u), _x1 = (xa1 + (xo)) ^ ((KK) * 64u); \
        F.w00 = lds_rdtr8<(KK) * 16384 + 0>(_w); F.w01 = lds_rdtr8<(KK) * 16384 + 2048>(_w); \
        F.w02 = lds_rdtr8<(KK) * 16384 + 4096>(_w); F.w03 = lds_rdtr8<(KK) * 16384 + 6144>(_w); \
        F.x00 = lds_rd128<0>(_x0); F.x01 = lds_rd128<0>(_x1); } while (0)
#define RD_G1(F, KK, xo) do { const unsigned _x0 = (xa0 + (xo)) ^ ((KK) * 64u), _x1 = (xa1 + (xo)) ^ ((KK) * 64u); \
        F.x10 = lds_rd128<4096>(_x0); F.x11 = lds_rd128<4096>(_x1); } while (0)
#define RD_G2(F, KK, wo) do { const unsigned _w = (wa0 + (wo)) ^ 32u; \
        F.w10 = lds_rdtr8<(KK) * 16384 + 0>(_w); F.w11 = lds_rdtr8<(KK) * 16384 + 2048>(_w); \
        F.w12 = lds_rdtr8<(KK) * 16384 + 4096>(_w); F.w13 = lds_rdtr8<(KK) * 16384 + 6144>(_w); } while (0)
#define MM00(F) mfma(acc00, cat8(F.x00, F.x01), cat8(F.w00, F.w01, F.w02, F.w03))
#define MM10(F) mfma(acc10, cat8(F.x10, F.x11), cat8(F.w00, F.w01, F.w02, F.w03))
#define MM01(F) mfma(acc01, cat8(F.x00, F.x01), cat8(F.w10, F.w11, F.w12, F.w13))
#define MM11(F) mfma(acc11, cat8(F.x10, F.x11), cat8(F.w10, F.w11, F.w12, F.w13))

    Frag fa, fb;
    wait_vm<nv(-1)>();
    asm volatile("" : "+v"(tmo_v));
    const bool bad = (unsigned)__builtin_amdgcn_readfirstlane(tmo_v) == call;
    bar();
    if (bad) {
        wait_vm<0>();
        float* o = OUT_PTR(lane);
        for (int r = 0; r < 32; ++r) {
            o[(size_t)((r & 3) + 8 * (r >> 2)) * NSH] = qn;
            o[(size_t)((r & 3) + 8 * (r >> 2)) * NSH + 32] = qn;
        }
        return;
    }
    RD_G0(fa, 0, 0u, 0u); RD_G1(fa, 0, 0u); RD_G2(fa, 0, 0u);
    wait_lgkm<0>();
    SCHED_FENCE();

    unsigned xo = 0, wo = 0;

#define STAGE(S, NV, IW, IX) do { \
        const unsigned xon = (xo + XSLOT) & (XR * XSLOT - 1); \
        const unsigned won = (wo == (WR - 1) * WSLOT) ? 0u : wo + WSLOT; \
        RD_G0(fb, 1, xo, wo); \
        SCHED_FENCE(); MM00(fa); SCHED_FENCE(); \
        RD_G1(fb, 1, xo); RD_G2(fb, 1, wo); \
        SCHED_FENCE(); MM10(fa); SCHED_FENCE(); \
        MM01(fa); SCHED_FENCE(); \
        MM11(fa); SCHED_FENCE(); \
        wait_vm<NV>(); \
        wait_lgkm<0>(); \
        bar(); \
        SCHED_FENCE(); \
        MM00(fb); SCHED_FENCE(); \
        RD_G0(fa, 0, xon, won); \
        if (IW) ISSUE_W01((S) + WR, wo); \
        SCHED_FENCE(); MM10(fb); SCHED_FENCE(); \
        RD_G1(fa, 0, xon); RD_G2(fa, 0, won); \
        if (IW) ISSUE_W23((S) + WR, wo); \
        SCHED_FENCE(); MM01(fb); SCHED_FENCE(); \
        if (IX) ISSUE_X((S) + XR, xo); \
        SCHED_FENCE(); MM11(fb); SCHED_FENCE(); \
        wait_lgkm<0>(); \
        SCHED_FENCE(); \
        xo = xon; wo = won; } while (0)
#define STAGE_C(S) STAGE(S, nv(S), (S) + WR < NSTG, (S) + XR < NSTG)

    constexpr int A0 = 2, E0 = NSTG - 4;
    static_assert(nv_const(A0, E0) && E0 + XR <= NSTG, "stage schedule");
    STAGE_C(0); STAGE_C(1);
    for (int S = A0; S < E0; ++S) STAGE(S, nv(A0), true, true);
    STAGE_C(28); STAGE_C(29); STAGE_C(30);
    RD_G0(fb, 1, xo, wo); RD_G1(fb, 1, xo); RD_G2(fb, 1, wo);
    SCHED_FENCE();
    MM00(fa); MM10(fa); MM01(fa); MM11(fa);
    SCHED_FENCE();
    wait_lgkm<0>();
    SCHED_FENCE();
    MM00(fb); MM10(fb); MM01(fb); MM11(fb);
    asm volatile("s_nop 15\n\ts_nop 15" : "+v"(acc00), "+v"(acc01), "+v"(acc10), "+v"(acc11));
    STAMP(3);

    {
        int lz = lane;
        asm volatile("" : "+v"(lz));
        float* o = OUT_PTR(lz);
#pragma unroll
        for (int r = 0; r < 16; ++r) {
            const size_t ro = (size_t)((r & 3) + 8 * (r >> 2)) * NSH;
            const float y00 = acc00[r] * scale, y01 = acc01[r] * scale, y10 = acc10[r] * scale, y11 = acc11[r] * scale;
            o[ro] = y00 < 0.f ? 0.f : y00;
            o[ro + 32] = y01 < 0.f ? 0.f : y01;
            o[ro + (size_t)32 * NSH] = y10 < 0.f ? 0.f : y10;
            o[ro + (size_t)32 * NSH + 32] = y11 < 0.f ? 0.f : y11;
        }
    }
}

}

extern "C" void kernel_launch(int rank, int nranks, ncclComm_t comm,
                              hipStream_t stream,
                              void* const* d_in, const int* in_sizes, int n_in,
                              void* d_out, int out_size,
                              const int* in_dtypes, int out_dtype,
                              void* const* d_out_all,
                              void* const* d_ws_all, size_t ws_bytes) {
    (void)comm; (void)in_dtypes; (void)out_dtype; (void)d_out_all;
    if (nranks != ag::NR || n_in < 4 || ws_bytes < ag::WS_NEED || (unsigned)rank >= (unsigned)ag::NR ||
        in_sizes[0] != ag::MSH * ag::KD || in_sizes[1] != ag::KD * ag::NSH || out_size != ag::NR * ag::MSH * ag::NSH) {
        fprintf(stderr, "kernel_launch: unexpected problem shape\n");
        return;
    }
    {
        static bool attr_set[64];
        int dev = 0;
        HIP_CHECK(hipGetDevice(&dev));
        if ((unsigned)dev >= 64u || !attr_set[dev]) {
            HIP_CHECK(hipFuncSetAttribute((const void*)ag::ag_gemm_kernel, hipFuncAttributeMaxDynamicSharedMemorySize, (int)ag::LDS_BYTES));
            if ((unsigned)dev < 64u) attr_set[dev] = true;
        }
    }
    ag::Params p;
    p.x = (const uint8_t*)d_in[0];
    p.w = (const uint8_t*)d_in[1];
    p.sx = (const float*)d_in[2];
    p.sw = (const float*)d_in[3];
    p.out = (float*)d_out;
    for (int i = 0; i < ag::NR; ++i) p.ws[i] = (uint8_t*)d_ws_all[i];
    p.myws = (uint8_t*)d_ws_all[rank];
    p.rank = rank;
    hipLaunchKernelGGL(ag::ag_gemm_kernel, dim3(ag::NWG), dim3(ag::NTHR), ag::LDS_BYTES, stream, p);
    HIP_CHECK(hipGetLastError());
}
```

```cpp
#include <hip/hip_runtime.h>
#include <rccl/rccl.h>
#include <cstdio>
#include <cstdint>

#define HIP_CHECK(expr) do { hipError_t _e = (expr); if (_e != hipSuccess) { \
    fprintf(stderr, "HIP error %s at %s:%d\n", hipGetErrorString(_e), __FILE__, __LINE__); \
    return; } } while (0)

namespace ag {

#define LAS __attribute__((address_space(3)))
typedef int v8i __attribute__((ext_vector_type(8)));
typedef int v4i __attribute__((ext_vector_type(4)));
typedef int v2i __attribute__((ext_vector_type(2)));
typedef unsigned v4u __attribute__((ext_vector_type(4)));
typedef float v16f __attribute__((ext_vector_type(16)));
typedef float v4f __attribute__((ext_vector_type(4)));

constexpr int NR = 8;
constexpr int MSH = 512;
constexpr int KD = 4096;
constexpr int NSH = 2048;
constexpr int NSTG = KD / 128;
constexpr int NWG = 256;
constexpr int NTHR = 512;
constexpr unsigned XSLOT = 16384;
constexpr unsigned WSLOT = 32768;
constexpr int XR = 4, WR = 3;
constexpr unsigned WOFF = XR * XSLOT;
constexpr unsigned LDS_BYTES = WOFF + WR * WSLOT;

constexpr int nv(int S) {
    int n = 0, last = 0;
    for (int i = 0; i < WR; ++i) { n += 4; if (i == S + 1) last = n; }
    for (int i = 0; i < XR; ++i) { n += 2; if (i == S + 1) last = n; }
    for (int s = 0; s < S; ++s) {
        if (s + WR < NSTG) { n += 4; if (s + WR == S + 1) last = n; }
        if (s + XR < NSTG) { n += 2; if (s + XR == S + 1) last = n; }
    }
    return n - last;
}
constexpr bool nv_const(int a, int b) { for (int s = a; s < b; ++s) if (nv(s) != nv(a)) return false; return true; }

constexpr size_t STAG_BYTES = (size_t)MSH * KD;
constexpr size_t OFF_FLAGS = 2 * STAG_BYTES;
constexpr size_t OFF_CTR = OFF_FLAGS + NR * 4 * 128;
constexpr size_t OFF_TMO = OFF_CTR + NWG * 4;
constexpr size_t OFF_STAMP = OFF_TMO + 256;
constexpr size_t WS_NEED = OFF_STAMP + NWG * 16 * 8;
constexpr unsigned long long DEADLINE_TICKS = 30000000ull;
#define PIECE(rb, kc) ((kc) * 4 + (rb))
#define AG_NE 8
#define AG_LATE 32
constexpr int NE = AG_NE;
constexpr int I0 = NE - XR - WR;
static_assert(NE == 32 || (I0 >= 0 && I0 + WR - 1 <= NE - XR - 1), "flag schedule");
#define AG_SKEW 0

struct Params {
    const uint8_t* x;
    const uint8_t* w;
    const float* sx;
    const float* sw;
    float* out;
    uint8_t* myws;
    uint8_t* ws[NR];
    int rank;
};

template <int N> __device__ __forceinline__ void wait_vm() { asm volatile("s_waitcnt vmcnt(%0)" ::"i"(N) : "memory"); }
template <int N> __device__ __forceinline__ void wait_lgkm() { asm volatile("s_waitcnt lgkmcnt(%0)" ::"i"(N) : "memory"); }
__device__ __forceinline__ void bar() {
    asm volatile("" ::: "memory");
    __builtin_amdgcn_s_barrier();
    asm volatile("" ::: "memory");
}
__device__ __forceinline__ void store16_sys(uint8_t* p, v4u v) {
    asm volatile("global_store_dwordx4 %0, %1, off sc0 sc1\n\ts_nop 1" ::"v"(p), "v"(v) : "memory");
}
__device__ __forceinline__ void store4_sys(unsigned* p, unsigned v) {
    asm volatile("global_store_dword %0, %1, off sc0 sc1" ::"v"(p), "v"(v) : "memory");
}
__device__ __forceinline__ unsigned long long rt() { return __builtin_amdgcn_s_memrealtime(); }
template <int OFF> __device__ __forceinline__ v4i lds_rd128(unsigned a) {
    v4i r; asm volatile("ds_read_b128 %0, %1 offset:%2" : "=&v"(r) : "v"(a), "i"(OFF) : "memory"); return r;
}
template <int OFF> __device__ __forceinline__ v2i lds_rdtr8(unsigned a) {
    v2i r; asm volatile("ds_read_b64_tr_b8 %0, %1 offset:%2" : "=&v"(r) : "v"(a), "i"(OFF) : "memory"); return r;
}
__device__ __forceinline__ v8i cat8(v4i a, v4i b) { return __builtin_shufflevector(a, b, 0, 1, 2, 3, 4, 5, 6, 7); }
__device__ __forceinline__ v8i cat8(v2i a, v2i b, v2i c, v2i d) {
    return cat8(__builtin_shufflevector(a, b, 0, 1, 2, 3), __builtin_shufflevector(c, d, 0, 1, 2, 3));
}
__device__ __forceinline__ void mfma(v16f& c, v8i x, v8i w) {
    asm volatile("v_mfma_f32_32x32x64_f8f6f4 %0, %1, %2, %0 cbsz:1 blgp:1" : "+v"(c) : "v"(x), "v"(w));
}
#define SCHED_FENCE() __builtin_amdgcn_sched_barrier(0)
#define AG_ST 1
__device__ __forceinline__ void st_out(float* p, float v) {
    __builtin_nontemporal_store(v, p);
}

#define STAMP(i) do { } while (0)

struct Frag {
    v4i x00, x01, x10, x11;
    v2i w00, w01, w02, w03, w10, w11, w12, w13;
};

__global__ __launch_bounds__(NTHR, 2) void ag_gemm_kernel(Params p) {
    extern __shared__ __attribute__((aligned(16))) uint8_t lds_raw[];
    LAS uint8_t* lds = (LAS uint8_t*)lds_raw;
    const int tid = threadIdx.x;
    const int lane = tid & 63;
    const int wid = __builtin_amdgcn_readfirstlane(tid >> 6);
    const int b = blockIdx.x;
    const int g = b & 7, idx = b >> 3;
    const int rank = p.rank;
    uint8_t* myws = p.myws;
    STAMP(0);

    unsigned* ctr = (unsigned*)(myws + OFF_CTR) + b;
    unsigned* tmo = (unsigned*)(myws + OFF_TMO);
    const float sx_v = p.sx[0], sw_v = p.sw[0];
    unsigned ctr_v;
    unsigned call;

    if (b < 128 && wid < 4) {
        const int crb = b >> 5, ckc = b & 31;
        const uint8_t* src = p.x + (size_t)crb * 128 * KD + ckc * 128;
        v4u d[4];
        const int cu0 = wid * 256 + lane;
#pragma unroll
        for (int i = 0; i < 4; ++i) {
            const int r = (cu0 >> 3) + 8 * i;
            d[i] = *(const v4u*)(src + (unsigned)(r * KD + (((lane & 7) ^ ((r >> 1) & 7)) << 4)));
        }
        asm volatile("global_load_dword %0, %1, off sc1" : "=v"(ctr_v) : "v"(ctr) : "memory");
        unsigned* f = (unsigned*)(p.ws[lane & 7] + OFF_FLAGS) + (rank * 4 + crb) * 32 + ckc;
        asm volatile("s_waitcnt vmcnt(0)" : "+v"(ctr_v), "+v"(d[0]), "+v"(d[1]), "+v"(d[2]), "+v"(d[3]), "+v"(f)::"memory");
        call = __builtin_amdgcn_readfirstlane(ctr_v) + 1u;
        STAMP(6);
        if (ckc >= NE) __builtin_amdgcn_s_sleep(AG_LATE);
        uint8_t* dst = myws + (size_t)(call & 1u) * STAG_BYTES + (size_t)PIECE(crb, ckc) * 16384 + cu0 * 16;
#pragma unroll
        for (int i = 0; i < 4; ++i) store16_sys(dst + 1024 * i, d[i]);
        wait_vm<0>();
        bar();
        if (wid == 0 && lane < NR && lane != rank) store4_sys(f, call);
    } else
    {
        asm volatile("global_load_dword %0, %1, off sc1" : "=v"(ctr_v) : "v"(ctr) : "memory");
        if (b < 128) bar();
        asm volatile("s_waitcnt vmcnt(0)" : "+v"(ctr_v)::"memory");
        call = __builtin_amdgcn_readfirstlane(ctr_v) + 1u;
        STAMP(6);
    }
    float scale = sx_v * sw_v;
    asm volatile("" : "+v"(scale));
    STAMP(1);

    const unsigned peer = (unsigned)(rank + g) & 7u;
    const int rb = idx >> 3, cb = idx & 7;
    const bool local = (g == 0);

    const int u0 = wid * 128 + lane;
    const int r0 = u0 >> 3, r1 = r0 + 8, pos = lane & 7;
    const uint8_t* aBase;
    unsigned aoff0, aoff1, aStride;
    if (local) {
        aBase = p.x + (size_t)rb * 128 * KD;
        aoff0 = (unsigned)(r0 * KD + ((pos ^ ((r0 >> 1) & 7)) << 4));
        aoff1 = (unsigned)(r1 * KD + ((pos ^ ((r1 >> 1) & 7)) << 4));
        aStride = 128u;
    } else {
        aBase = p.ws[peer] + (size_t)(call & 1u) * STAG_BYTES + (size_t)PIECE(rb, 0) * 16384;
        aoff0 = (unsigned)u0 * 16u; aoff1 = aoff0 + 1024u;
        aStride = (PIECE(0, 1) - PIECE(0, 0)) * 16384u;
    }
    const int wr0 = wid * 16 + (lane >> 4);
    const unsigned boff0 = (unsigned)((wr0 + 0) * NSH + (((lane & 15) ^ (((wr0 + 0) & 7) << 1)) << 4));
    const unsigned boff1 = (unsigned)((wr0 + 4) * NSH + (((lane & 15) ^ (((wr0 + 4) & 7) << 1)) << 4));
    const uint8_t* bBase = p.w + cb * 256;
    const unsigned ldsx = (unsigned)wid * 2048u, ldsw = WOFF + (unsigned)wid * 4096u;

#define GLDS(gp, lo) __builtin_amdgcn_global_load_lds((const unsigned*)(gp), (LAS unsigned*)(lds + (lo)), 16, 0, 0)
#define ISSUE_X(S, xo) do { const uint8_t* _a = aBase + (size_t)(S) * aStride; \
        GLDS(_a + aoff0, (xo) + ldsx); GLDS(_a + aoff1, (xo) + ldsx + 1024); } while (0)
#define ISSUE_W01(S, wo) do { const uint8_t* _b = bBase + (size_t)(S) * (128 * NSH); \
        GLDS(_b + boff0, (wo) + ldsw); GLDS(_b + boff1, (wo) + ldsw + 1024); } while (0)
#define ISSUE_W23(S, wo) do { const uint8_t* _b = bBase + (size_t)(S) * (128 * NSH) + 8 * NSH; \
        GLDS(_b + boff0, (wo) + ldsw + 2048); GLDS(_b + boff1, (wo) + ldsw + 3072); } while (0)

#pragma unroll
    for (int s = 0; s < WR; ++s) { ISSUE_W01(s, s * WSLOT); ISSUE_W23(s, s * WSLOT); }

    const unsigned* fl = (lane < 32) ? (const unsigned*)(myws + OFF_FLAGS) + (peer * 4 + rb) * 32 + lane : (const unsigned*)tmo;
    auto wait_flags = [&](int n) -> bool {
        const unsigned long long t0 = rt();
        for (unsigned it = 1;; ++it) {
            const unsigned v = __hip_atomic_load(fl, __ATOMIC_RELAXED, __HIP_MEMORY_SCOPE_SYSTEM);
            if (__any(lane >= 32 && v == call)) return false;
            if (__all(lane >= n || (int)(v - call) >= 0)) return true;
            if ((it & 15u) == 0u && rt() - t0 > DEADLINE_TICKS) {
                if (lane == 0) __hip_atomic_store(tmo, call, __ATOMIC_RELAXED, __HIP_MEMORY_SCOPE_AGENT);
                return false;
            }
            __builtin_amdgcn_s_sleep(1);
        }
    };
    if (!local && wid == 0) {
        (void)wait_flags(NE);
        for (int i = (AG_SKEW >> (4 * rb)) & 15; i > 0; --i) __builtin_amdgcn_s_sleep(8);
        wait_vm<0>();
    }
    bar();
    STAMP(2);
    if (tid == 0) __hip_atomic_store(ctr, call, __ATOMIC_RELAXED, __HIP_MEMORY_SCOPE_AGENT);
    unsigned tmo_v;
    asm volatile("global_load_dword %0, %1, off sc1" : "=v"(tmo_v) : "v"(tmo) : "memory");

#pragma unroll
    for (int s = 0; s < XR; ++s) ISSUE_X(s, s * XSLOT);

    const int wm = wid >> 2, wn = wid & 3;
    const int h = lane >> 5;
    const float qn = __builtin_nanf("");
#define OUT_PTR(L) (p.out + ((size_t)peer * MSH + rb * 128 + wm * 64 + 4 * ((L) >> 5)) * NSH + cb * 256 + wn * 64 + ((L) & 31))

    const unsigned ldsBase = (unsigned)(uintptr_t)lds;
    const int sw = ((lane & 31) >> 1) & 7;
    const unsigned xrow = ldsBase + (unsigned)(wm * 64 + (lane & 31)) * 128u;
    const unsigned xa0 = xrow + (unsigned)(((2 * h + 0) ^ sw) << 4);
    const unsigned xa1 = xrow + (unsigned)(((2 * h + 1) ^ sw) << 4);
    const int g16 = lane >> 4, i16 = lane & 15, q8 = i16 >> 1;
    const unsigned wa0 = ldsBase + WOFF + (unsigned)(32 * h + q8) * 256u +
                         (unsigned)(((wn * 4 + (g16 & 1)) ^ (q8 << 1)) << 4) + (unsigned)(i16 & 1) * 8u;

    v16f acc00, acc01, acc10, acc11;
#pragma unroll
    for (int i = 0; i < 16; ++i) { acc00[i] = 0.f; acc01[i] = 0.f; acc10[i] = 0.f; acc11[i] = 0.f; }
    asm volatile("s_nop 1" : "+v"(acc00), "+v"(acc01), "+v"(acc10), "+v"(acc11));

#define RD_G0(F, KK, xo, wo) do { const unsigned _w = wa0 + (wo), _x0 = (xa0 + (xo)) ^ ((KK) * 64u), _x1 = (xa1 + (xo)) ^ ((KK) * 64u); \
        F.w00 = lds_rdtr8<(KK) * 16384 + 0>(_w); F.w01 = lds_rdtr8<(KK) * 16384 + 2048>(_w); \
        F.w02 = lds_rdtr8<(KK) * 16384 + 4096>(_w); F.w03 = lds_rdtr8<(KK) * 16384 + 6144>(_w); \
        F.x00 = lds_rd128<0>(_x0); F.x01 = lds_rd128<0>(_x1); } while (0)
#define RD_G1(F, KK, xo) do { const unsigned _x0 = (xa0 + (xo)) ^ ((KK) * 64u), _x1 = (xa1 + (xo)) ^ ((KK) * 64u); \
        F.x10 = lds_rd128<4096>(_x0); F.x11 = lds_rd128<4096>(_x1); } while (0)
#define RD_G2(F, KK, wo) do { const unsigned _w = (wa0 + (wo)) ^ 32u; \
        F.w10 = lds_rdtr8<(KK) * 16384 + 0>(_w); F.w11 = lds_rdtr8<(KK) * 16384 + 2048>(_w); \
        F.w12 = lds_rdtr8<(KK) * 16384 + 4096>(_w); F.w13 = lds_rdtr8<(KK) * 16384 + 6144>(_w); } while (0)
#define MM00(F) mfma(acc00, cat8(F.x00, F.x01), cat8(F.w00, F.w01, F.w02, F.w03))
#define MM10(F) mfma(acc10, cat8(F.x10, F.x11), cat8(F.w00, F.w01, F.w02, F.w03))
#define MM01(F) mfma(acc01, cat8(F.x00, F.x01), cat8(F.w10, F.w11, F.w12, F.w13))
#define MM11(F) mfma(acc11, cat8(F.x10, F.x11), cat8(F.w10, F.w11, F.w12, F.w13))

    Frag fa, fb;
    wait_vm<nv(-1)>();
    asm volatile("" : "+v"(tmo_v));
    bool bad = (unsigned)__builtin_amdgcn_readfirstlane(tmo_v) == call;
    bar();
    if (bad) {
        wait_vm<0>();
        float* o = OUT_PTR(lane);
        for (int r = 0; r < 32; ++r) {
            o[(size_t)((r & 3) + 8 * (r >> 2)) * NSH] = qn;
            o[(size_t)((r & 3) + 8 * (r >> 2)) * NSH + 32] = qn;
        }
        return;
    }
    RD_G0(fa, 0, 0u, 0u); RD_G1(fa, 0, 0u); RD_G2(fa, 0, 0u);
    wait_lgkm<0>();
    SCHED_FENCE();

    unsigned xo = 0, wo = 0;
    unsigned flv = 0;

#define STAGE(S, NV, IW, IX) do { \
        const unsigned xon = (xo + XSLOT) & (XR * XSLOT - 1); \
        const unsigned won = (wo == (WR - 1) * WSLOT) ? 0u : wo + WSLOT; \
        RD_G0(fb, 1, xo, wo); \
        SCHED_FENCE(); MM00(fa); SCHED_FENCE(); \
        RD_G1(fb, 1, xo); RD_G2(fb, 1, wo); \
        SCHED_FENCE(); MM10(fa); SCHED_FENCE(); \
        MM01(fa); SCHED_FENCE(); \
        MM11(fa); SCHED_FENCE(); \
        wait_vm<NV>(); \
        wait_lgkm<0>(); \
        bar(); \
        SCHED_FENCE(); \
        if (NE < 32 && (S) == I0 + WR - 1 && !local) { \
            asm volatile("" : "+v"(flv)); \
            if (__any(lane >= 32 && flv == call)) bad = true; \
            else if (!__all(lane >= 32 || (int)(flv - call) >= 0)) bad = !wait_flags(32); \
        } \
        if (NE < 32 && (S) == I0) asm volatile("global_load_dword %0, %1, off sc0 sc1" : "=v"(flv) : "v"(fl) : "memory"); \
        MM00(fb); SCHED_FENCE(); \
        RD_G0(fa, 0, xon, won); \
        if (IW) ISSUE_W01((S) + WR, wo); \
        SCHED_FENCE(); MM10(fb); SCHED_FENCE(); \
        RD_G1(fa, 0, xon); RD_G2(fa, 0, won); \
        if (IW) ISSUE_W23((S) + WR, wo); \
        SCHED_FENCE(); MM01(fb); SCHED_FENCE(); \
        if (IX) ISSUE_X((S) + XR, xo); \
        SCHED_FENCE(); MM11(fb); SCHED_FENCE(); \
        wait_lgkm<0>(); \
        SCHED_FENCE(); \
        xo = xon; wo = won; } while (0)
#define STAGE_C(S) STAGE(S, nv(S), (S) + WR < NSTG, (S) + XR < NSTG)
#define STAGE_L(S, NV) do { constexpr int I0 = -100; STAGE(S, NV, true, true); } while (0)

    constexpr int A0 = 4, E0 = NSTG - 4;
    static_assert(nv_const(A0, E0) && E0 + XR <= NSTG && (NE == 32 || I0 + WR - 1 < A0), "stage schedule");
    STAGE_C(0); STAGE_C(1); STAGE_C(2); STAGE_C(3);
    for (int S = A0; S < E0; ++S) STAGE_L(S, nv(A0));
    STAGE_C(28); STAGE_C(29); STAGE_C(30);
    RD_G0(fb, 1, xo, wo); RD_G1(fb, 1, xo); RD_G2(fb, 1, wo);
    SCHED_FENCE();
    MM00(fa); MM10(fa); MM01(fa); MM11(fa);
    SCHED_FENCE();
    wait_lgkm<0>();
    SCHED_FENCE();
    MM00(fb); MM10(fb); MM01(fb); MM11(fb);
    asm volatile("s_nop 15\n\ts_nop 15" : "+v"(acc00), "+v"(acc01), "+v"(acc10), "+v"(acc11));
    STAMP(3);

    {
        int lz = lane;
        asm volatile("" : "+v"(lz));
        float* o = OUT_PTR(lz);
        if (bad) scale = qn;
#pragma unroll
        for (int r = 0; r < 16; ++r) {
            const size_t ro = (size_t)((r & 3) + 8 * (r >> 2)) * NSH;
            const float y00 = acc00[r] * scale, y01 = acc01[r] * scale, y10 = acc10[r] * scale, y11 = acc11[r] * scale;
            st_out(o + ro, y00 < 0.f ? 0.f : y00);
            st_out(o + ro + 32, y01 < 0.f ? 0.f : y01);
            st_out(o + ro + (size_t)32 * NSH, y10 < 0.f ? 0.f : y10);
            st_out(o + ro + (size_t)32 * NSH + 32, y11 < 0.f ? 0.f : y11);
        }
    }
}

}

extern "C" void kernel_launch(int rank, int nranks, ncclComm_t comm,
                              hipStream_t stream,
                              void* const* d_in, const int* in_sizes, int n_in,
                              void* d_out, int out_size,
                              const int* in_dtypes, int out_dtype,
                              void* const* d_out_all,
                              void* const* d_ws_all, size_t ws_bytes) {
    (void)comm; (void)in_dtypes; (void)out_dtype; (void)d_out_all;
    if (nranks != ag::NR || n_in < 4 || ws_bytes < ag::WS_NEED || (unsigned)rank >= (unsigned)ag::NR ||
        in_sizes[0] != ag::MSH * ag::KD || in_sizes[1] != ag::KD * ag::NSH || out_size != ag::NR * ag::MSH * ag::NSH) {
        fprintf(stderr, "kernel_launch: unexpected problem shape\n");
        return;
    }
    {
        static bool attr_set[64];
        int dev = 0;
        HIP_CHECK(hipGetDevice(&dev));
        if ((unsigned)dev >= 64u || !attr_set[dev]) {
            HIP_CHECK(hipFuncSetAttribute((const void*)ag::ag_gemm_kernel, hipFuncAttributeMaxDynamicSharedMemorySize, (int)ag::LDS_BYTES));
            if ((unsigned)dev < 64u) attr_set[dev] = true;
        }
    }
    ag::Params p;
    p.x = (const uint8_t*)d_in[0];
    p.w = (const uint8_t*)d_in[1];
    p.sx = (const float*)d_in[2];
    p.sw = (const float*)d_in[3];
    p.out = (float*)d_out;
    for (int i = 0; i < ag::NR; ++i) p.ws[i] = (uint8_t*)d_ws_all[i];
    p.myws = (uint8_t*)d_ws_all[rank];
    p.rank = rank;
    hipLaunchKernelGGL(ag::ag_gemm_kernel, dim3(ag::NWG), dim3(ag::NTHR), ag::LDS_BYTES, stream, p);
    HIP_CHECK(hipGetLastError());
}
```

```cpp
#include <hip/hip_runtime.h>
#include <rccl/rccl.h>
#include <cstdio>
#include <cstdint>

#define HIP_CHECK(expr) do { hipError_t _e = (expr); if (_e != hipSuccess) { \
    fprintf(stderr, "HIP error %s at %s:%d\n", hipGetErrorString(_e), __FILE__, __LINE__); \
    return; } } while (0)

namespace ag {

#define LAS __attribute__((address_space(3)))
typedef int v8i __attribute__((ext_vector_type(8)));
typedef int v4i __attribute__((ext_vector_type(4)));
typedef int v2i __attribute__((ext_vector_type(2)));
typedef unsigned v4u __attribute__((ext_vector_type(4)));
typedef float v16f __attribute__((ext_vector_type(16)));
typedef float v4f __attribute__((ext_vector_type(4)));

constexpr int NR = 8;
constexpr int MSH = 512;
constexpr int KD = 4096;
constexpr int NSH = 2048;
constexpr int NSTG = KD / 128;
constexpr int NWG = 256;
constexpr int NTHR = 512;
constexpr unsigned XSLOT = 16384;
constexpr unsigned WSLOT = 32768;
constexpr int XR = 4, WR = 3;
constexpr unsigned WOFF = XR * XSLOT;
constexpr unsigned LDS_BYTES = WOFF + WR * WSLOT;

constexpr int nv(int S) {
    int n = 0, last = 0;
    for (int i = 0; i < WR; ++i) { n += 4; if (i == S + 1) last = n; }
    for (int i = 0; i < XR; ++i) { n += 2; if (i == S + 1) last = n; }
    for (int s = 0; s < S; ++s) {
        if (s + WR < NSTG) { n += 4; if (s + WR == S + 1) last = n; }
        if (s + XR < NSTG) { n += 2; if (s + XR == S + 1) last = n; }
    }
    return n - last;
}
constexpr bool nv_const(int a, int b) { for (int s = a; s < b; ++s) if (nv(s) != nv(a)) return false; return true; }

constexpr size_t STAG_BYTES = (size_t)MSH * KD;
constexpr size_t OFF_FLAGS = 2 * STAG_BYTES;
constexpr size_t OFF_CTR = OFF_FLAGS + NR * 4 * 128;
constexpr size_t OFF_TMO = OFF_CTR + NWG * 4;
constexpr size_t OFF_STAMP = OFF_TMO + 256;
constexpr size_t WS_NEED = OFF_STAMP + NWG * 16 * 8;
constexpr unsigned long long DEADLINE_TICKS = 30000000ull;
#define PIECE(rb, kc) ((kc) * 4 + (rb))
constexpr int LATE = 32;
constexpr int NE = 8;
constexpr int I0 = NE - XR - WR;
static_assert(I0 >= 0 && I0 + WR - 1 <= NE - XR - 1, "flag schedule");

struct Params {
    const uint8_t* x;
    const uint8_t* w;
    const float* sx;
    const float* sw;
    float* out;
    uint8_t* myws;
    uint8_t* ws[NR];
    int rank;
};

template <int N> __device__ __forceinline__ void wait_vm() { asm volatile("s_waitcnt vmcnt(%0)" ::"i"(N) : "memory"); }
template <int N> __device__ __forceinline__ void wait_lgkm() { asm volatile("s_waitcnt lgkmcnt(%0)" ::"i"(N) : "memory"); }
__device__ __forceinline__ void bar() {
    asm volatile("" ::: "memory");
    __builtin_amdgcn_s_barrier();
    asm volatile("" ::: "memory");
}
__device__ __forceinline__ void store16_sys(uint8_t* p, v4u v) {
    asm volatile("global_store_dwordx4 %0, %1, off sc0 sc1\n\ts_nop 1" ::"v"(p), "v"(v) : "memory");
}
__device__ __forceinline__ void store4_sys(unsigned* p, unsigned v) {
    asm volatile("global_store_dword %0, %1, off sc0 sc1" ::"v"(p), "v"(v) : "memory");
}
__device__ __forceinline__ unsigned long long rt() { return __builtin_amdgcn_s_memrealtime(); }
template <int OFF> __device__ __forceinline__ v4i lds_rd128(unsigned a) {
    v4i r; asm volatile("ds_read_b128 %0, %1 offset:%2" : "=&v"(r) : "v"(a), "i"(OFF) : "memory"); return r;
}
template <int OFF> __device__ __forceinline__ v2i lds_rdtr8(unsigned a) {
    v2i r; asm volatile("ds_read_b64_tr_b8 %0, %1 offset:%2" : "=&v"(r) : "v"(a), "i"(OFF) : "memory"); return r;
}
__device__ __forceinline__ v8i cat8(v4i a, v4i b) { return __builtin_shufflevector(a, b, 0, 1, 2, 3, 4, 5, 6, 7); }
__device__ __forceinline__ v8i cat8(v2i a, v2i b, v2i c, v2i d) {
    return cat8(__builtin_shufflevector(a, b, 0, 1, 2, 3), __builtin_shufflevector(c, d, 0, 1, 2, 3));
}
__device__ __forceinline__ void mfma(v16f& c, v8i x, v8i w) {
    asm volatile("v_mfma_f32_32x32x64_f8f6f4 %0, %1, %2, %0 cbsz:1 blgp:1" : "+v"(c) : "v"(x), "v"(w));
}
#define SCHED_FENCE() __builtin_amdgcn_sched_barrier(0)
__device__ __forceinline__ void st_out(float* p, float v) { __builtin_nontemporal_store(v, p); }

#define STAMP(i) do { } while (0)

struct Frag {
    v4i x00, x01, x10, x11;
    v2i w00, w01, w02, w03, w10, w11, w12, w13;
};

__global__ __launch_bounds__(NTHR, 2) void ag_gemm_kernel(Params p) {
    extern __shared__ __attribute__((aligned(16))) uint8_t lds_raw[];
    LAS uint8_t* lds = (LAS uint8_t*)lds_raw;
    const int tid = threadIdx.x;
    const int lane = tid & 63;
    const int wid = __builtin_amdgcn_readfirstlane(tid >> 6);
    const int b = blockIdx.x;
    const int g = b & 7, idx = b >> 3;
    const int rank = p.rank;
    uint8_t* myws = p.myws;
    STAMP(0);

    unsigned* ctr = (unsigned*)(myws + OFF_CTR) + b;
    unsigned* tmo = (unsigned*)(myws + OFF_TMO);
    const float sx_v = p.sx[0], sw_v = p.sw[0];
    unsigned ctr_v;
    unsigned call;

    if (b < 128 && wid < 4) {
        const int crb = b >> 5, ckc = b & 31;
        const uint8_t* src = p.x + (size_t)crb * 128 * KD + ckc * 128;
        v4u d[4];
        const int cu0 = wid * 256 + lane;
#pragma unroll
        for (int i = 0; i < 4; ++i) {
            const int r = (cu0 >> 3) + 8 * i;
            d[i] = *(const v4u*)(src + (unsigned)(r * KD + (((lane & 7) ^ ((r >> 1) & 7)) << 4)));
        }
        asm volatile("global_load_dword %0, %1, off sc1" : "=v"(ctr_v) : "v"(ctr) : "memory");
        unsigned* f = (unsigned*)(p.ws[lane & 7] + OFF_FLAGS) + (rank * 4 + crb) * 32 + ckc;
        asm volatile("s_waitcnt vmcnt(0)" : "+v"(ctr_v), "+v"(d[0]), "+v"(d[1]), "+v"(d[2]), "+v"(d[3]), "+v"(f)::"memory");
        call = __builtin_amdgcn_readfirstlane(ctr_v) + 1u;
        STAMP(6);
        if (ckc >= NE) __builtin_amdgcn_s_sleep(LATE);
        uint8_t* dst = myws + (size_t)(call & 1u) * STAG_BYTES + (size_t)PIECE(crb, ckc) * 16384 + cu0 * 16;
#pragma unroll
        for (int i = 0; i < 4; ++i) store16_sys(dst + 1024 * i, d[i]);
        wait_vm<0>();
        bar();
        if (wid == 0 && lane < NR && lane != rank) store4_sys(f, call);
    } else {
        asm volatile("global_load_dword %0, %1, off sc1" : "=v"(ctr_v) : "v"(ctr) : "memory");
        if (b < 128) bar();
        asm volatile("s_waitcnt vmcnt(0)" : "+v"(ctr_v)::"memory");
        call = __builtin_amdgcn_readfirstlane(ctr_v) + 1u;
        STAMP(6);
    }
    float scale = sx_v * sw_v;
    asm volatile("" : "+v"(scale));
    STAMP(1);

    const unsigned peer = (unsigned)(rank + g) & 7u;
    const int rb = idx >> 3, cb = idx & 7;
    const bool local = (g == 0);

    const int u0 = wid * 128 + lane;
    const int r0 = u0 >> 3, r1 = r0 + 8, pos = lane & 7;
    const uint8_t* aBase;
    unsigned aoff0, aoff1, aStride;
    if (local) {
        aBase = p.x + (size_t)rb * 128 * KD;
        aoff0 = (unsigned)(r0 * KD + ((pos ^ ((r0 >> 1) & 7)) << 4));
        aoff1 = (unsigned)(r1 * KD + ((pos ^ ((r1 >> 1) & 7)) << 4));
        aStride = 128u;
    } else {
        aBase = p.ws[peer] + (size_t)(call & 1u) * STAG_BYTES + (size_t)PIECE(rb, 0) * 16384;
        aoff0 = (unsigned)u0 * 16u; aoff1 = aoff0 + 1024u;
        aStride = (PIECE(0, 1) - PIECE(0, 0)) * 16384u;
    }
    const int wr0 = wid * 16 + (lane >> 4);
    const unsigned boff0 = (unsigned)((wr0 + 0) * NSH + (((lane & 15) ^ (((wr0 + 0) & 7) << 1)) << 4));
    const unsigned boff1 = (unsigned)((wr0 + 4) * NSH + (((lane & 15) ^ (((wr0 + 4) & 7) << 1)) << 4));
    const uint8_t* bBase = p.w + cb * 256;
    const unsigned ldsx = (unsigned)wid * 2048u, ldsw = WOFF + (unsigned)wid * 4096u;

#define GLDS(gp, lo) __builtin_amdgcn_global_load_lds((const unsigned*)(gp), (LAS unsigned*)(lds + (lo)), 16, 0, 0)
#define ISSUE_X(S, xo) do { const uint8_t* _a = aBase + (size_t)(S) * aStride; \
        GLDS(_a + aoff0, (xo) + ldsx); GLDS(_a + aoff1, (xo) + ldsx + 1024); } while (0)
#define ISSUE_W01(S, wo) do { const uint8_t* _b = bBase + (size_t)(S) * (128 * NSH); \
        GLDS(_b + boff0, (wo) + ldsw); GLDS(_b + boff1, (wo) + ldsw + 1024); } while (0)
#define ISSUE_W23(S, wo) do { const uint8_t* _b = bBase + (size_t)(S) * (128 * NSH) + 8 * NSH; \
        GLDS(_b + boff0, (wo) + ldsw + 2048); GLDS(_b + boff1, (wo) + ldsw + 3072); } while (0)

#pragma unroll
    for (int s = 0; s < WR; ++s) { ISSUE_W01(s, s * WSLOT); ISSUE_W23(s, s * WSLOT); }

    const unsigned* fl = (lane < 32) ? (const unsigned*)(myws + OFF_FLAGS) + (peer * 4 + rb) * 32 + lane : (const unsigned*)tmo;
    auto wait_flags = [&](int n) -> bool {
        const unsigned long long t0 = rt();
        for (unsigned it = 1;; ++it) {
            const unsigned v = __hip_atomic_load(fl, __ATOMIC_RELAXED, __HIP_MEMORY_SCOPE_SYSTEM);
            if (__any(lane >= 32 && v == call)) return false;
            if (__all(lane >= n || (int)(v - call) >= 0)) return true;
            if ((it & 15u) == 0u && rt() - t0 > DEADLINE_TICKS) {
                if (lane == 0) __hip_atomic_store(tmo, call, __ATOMIC_RELAXED, __HIP_MEMORY_SCOPE_AGENT);
                return false;
            }
            __builtin_amdgcn_s_sleep(1);
        }
    };
    if (!local && wid == 0) {
        (void)wait_flags(NE);
        wait_vm<0>();
    }
    bar();
    STAMP(2);
    if (tid == 0) __hip_atomic_store(ctr, call, __ATOMIC_RELAXED, __HIP_MEMORY_SCOPE_AGENT);
    unsigned tmo_v;
    asm volatile("global_load_dword %0, %1, off sc1" : "=v"(tmo_v) : "v"(tmo) : "memory");

#pragma unroll
    for (int s = 0; s < XR; ++s) ISSUE_X(s, s * XSLOT);

    const int wm = wid >> 2, wn = wid & 3;
    const int h = lane >> 5;
    const float qn = __builtin_nanf("");
#define OUT_PTR(L) (p.out + ((size_t)peer * MSH + rb * 128 + wm * 64 + 4 * ((L) >> 5)) * NSH + cb * 256 + wn * 64 + ((L) & 31))

    const unsigned ldsBase = (unsigned)(uintptr_t)lds;
    const int sw = ((lane & 31) >> 1) & 7;
    const unsigned xrow = ldsBase + (unsigned)(wm * 64 + (lane & 31)) * 128u;
    const unsigned xa0 = xrow + (unsigned)(((2 * h + 0) ^ sw) << 4);
    const unsigned xa1 = xrow + (unsigned)(((2 * h + 1) ^ sw) << 4);
    const int g16 = lane >> 4, i16 = lane & 15, q8 = i16 >> 1;
    const unsigned wa0 = ldsBase + WOFF + (unsigned)(32 * h + q8) * 256u +
                         (unsigned)(((wn * 4 + (g16 & 1)) ^ (q8 << 1)) << 4) + (unsigned)(i16 & 1) * 8u;

    v16f acc00, acc01, acc10, acc11;
#pragma unroll
    for (int i = 0; i < 16; ++i) { acc00[i] = 0.f; acc01[i] = 0.f; acc10[i] = 0.f; acc11[i] = 0.f; }
    asm volatile("s_nop 1" : "+v"(acc00), "+v"(acc01), "+v"(acc10), "+v"(acc11));

#define RD_G0(F, KK, xo, wo) do { const unsigned _w = wa0 + (wo), _x0 = (xa0 + (xo)) ^ ((KK) * 64u), _x1 = (xa1 + (xo)) ^ ((KK) * 64u); \
        F.w00 = lds_rdtr8<(KK) * 16384 + 0>(_w); F.w01 = lds_rdtr8<(KK) * 16384 + 2048>(_w); \
        F.w02 = lds_rdtr8<(KK) * 16384 + 4096>(_w); F.w03 = lds_rdtr8<(KK) * 16384 + 6144>(_w); \
        F.x00 = lds_rd128<0>(_x0); F.x01 = lds_rd128<0>(_x1); } while (0)
#define RD_G1(F, KK, xo) do { const unsigned _x0 = (xa0 + (xo)) ^ ((KK) * 64u), _x1 = (xa1 + (xo)) ^ ((KK) * 64u); \
        F.x10 = lds_rd128<4096>(_x0); F.x11 = lds_rd128<4096>(_x1); } while (0)
#define RD_G2(F, KK, wo) do { const unsigned _w = (wa0 + (wo)) ^ 32u; \
        F.w10 = lds_rdtr8<(KK) * 16384 + 0>(_w); F.w11 = lds_rdtr8<(KK) * 16384 + 2048>(_w); \
        F.w12 = lds_rdtr8<(KK) * 16384 + 4096>(_w); F.w13 = lds_rdtr8<(KK) * 16384 + 6144>(_w); } while (0)
#define MM00(F) mfma(acc00, cat8(F.x00, F.x01), cat8(F.w00, F.w01, F.w02, F.w03))
#define MM10(F) mfma(acc10, cat8(F.x10, F.x11), cat8(F.w00, F.w01, F.w02, F.w03))
#define MM01(F) mfma(acc01, cat8(F.x00, F.x01), cat8(F.w10, F.w11, F.w12, F.w13))
#define MM11(F) mfma(acc11, cat8(F.x10, F.x11), cat8(F.w10, F.w11, F.w12, F.w13))

    Frag fa, fb;
    wait_vm<nv(-1)>();
    asm volatile("" : "+v"(tmo_v));
    bool bad = (unsigned)__builtin_amdgcn_readfirstlane(tmo_v) == call;
    bar();
    if (bad) {
        wait_vm<0>();
        float* o = OUT_PTR(lane);
        for (int r = 0; r < 32; ++r) {
            o[(size_t)((r & 3) + 8 * (r >> 2)) * NSH] = qn;
            o[(size_t)((r & 3) + 8 * (r >> 2)) * NSH + 32] = qn;
        }
        return;
    }
    RD_G0(fa, 0, 0u, 0u); RD_G1(fa, 0, 0u); RD_G2(fa, 0, 0u);
    wait_lgkm<0>();
    SCHED_FENCE();

    unsigned xo = 0, wo = 0;
    unsigned flv = 0;

#define STAGE(S, NV, IW, IX) do { \
        const unsigned xon = (xo + XSLOT) & (XR * XSLOT - 1); \
        const unsigned won = (wo == (WR - 1) * WSLOT) ? 0u : wo + WSLOT; \
        RD_G0(fb, 1, xo, wo); \
        SCHED_FENCE(); MM00(fa); SCHED_FENCE(); \
        RD_G1(fb, 1, xo); RD_G2(fb, 1, wo); \
        SCHED_FENCE(); MM10(fa); SCHED_FENCE(); \
        MM01(fa); SCHED_FENCE(); \
        MM11(fa); SCHED_FENCE(); \
        wait_vm<NV>(); \
        wait_lgkm<0>(); \
        bar(); \
        SCHED_FENCE(); \
        if ((S) == I0 + WR - 1 && !local) { \
            asm volatile("" : "+v"(flv)); \
            if (__any(lane >= 32 && flv == call)) bad = true; \
            else if (!__all(lane >= 32 || (int)(flv - call) >= 0)) bad = !wait_flags(32); \
        } \
        if ((S) == I0) asm volatile("global_load_dword %0, %1, off sc0 sc1" : "=v"(flv) : "v"(fl) : "memory"); \
        MM00(fb); SCHED_FENCE(); \
        RD_G0(fa, 0, xon, won); \
        if (IW) ISSUE_W01((S) + WR, wo); \
        SCHED_FENCE(); MM10(fb); SCHED_FENCE(); \
        RD_G1(fa, 0, xon); RD_G2(fa, 0, won); \
        if (IW) ISSUE_W23((S) + WR, wo); \
        SCHED_FENCE(); MM01(fb); SCHED_FENCE(); \
        if (IX) ISSUE_X((S) + XR, xo); \
        SCHED_FENCE(); MM11(fb); SCHED_FENCE(); \
        wait_lgkm<0>(); \
        SCHED_FENCE(); \
        xo = xon; wo = won; } while (0)
#define STAGE_C(S) STAGE(S, nv(S), (S) + WR < NSTG, (S) + XR < NSTG)
#define STAGE_L(S, NV) do { constexpr int I0 = -100; STAGE(S, NV, true, true); } while (0)

    constexpr int A0 = 4, E0 = NSTG - 4;
    static_assert(nv_const(A0, E0) && E0 + XR <= NSTG && I0 + WR - 1 < A0, "stage schedule");
    STAGE_C(0); STAGE_C(1); STAGE_C(2); STAGE_C(3);
    for (int S = A0; S < E0; ++S) STAGE_L(S, nv(A0));
    STAGE_C(28); STAGE_C(29); STAGE_C(30);
    RD_G0(fb, 1, xo, wo); RD_G1(fb, 1, xo); RD_G2(fb, 1, wo);
    SCHED_FENCE();
    MM00(fa); MM10(fa); MM01(fa); MM11(fa);
    SCHED_FENCE();
    wait_lgkm<0>();
    SCHED_FENCE();
    MM00(fb); MM10(fb); MM01(fb); MM11(fb);
    asm volatile("s_nop 15\n\ts_nop 15" : "+v"(acc00), "+v"(acc01), "+v"(acc10), "+v"(acc11));
    STAMP(3);

    {
        int lz = lane;
        asm volatile("" : "+v"(lz));
        float* o = OUT_PTR(lz);
        if (bad) scale = qn;
#pragma unroll
        for (int r = 0; r < 16; ++r) {
            const size_t ro = (size_t)((r & 3) + 8 * (r >> 2)) * NSH;
            const float y00 = acc00[r] * scale, y01 = acc01[r] * scale, y10 = acc10[r] * scale, y11 = acc11[r] * scale;
            st_out(o + ro, y00 < 0.f ? 0.f : y00);
            st_out(o + ro + 32, y01 < 0.f ? 0.f : y01);
            st_out(o + ro + (size_t)32 * NSH, y10 < 0.f ? 0.f : y10);
            st_out(o + ro + (size_t)32 * NSH + 32, y11 < 0.f ? 0.f : y11);
        }
    }
}

}

extern "C" void kernel_launch(int rank, int nranks, ncclComm_t comm,
                              hipStream_t stream,
                              void* const* d_in, const int* in_sizes, int n_in,
                              void* d_out, int out_size,
                              const int* in_dtypes, int out_dtype,
                              void* const* d_out_all,
                              void* const* d_ws_all, size_t ws_bytes) {
    (void)comm; (void)in_dtypes; (void)out_dtype; (void)d_out_all;
    if (nranks != ag::NR || n_in < 4 || ws_bytes < ag::WS_NEED || (unsigned)rank >= (unsigned)ag::NR ||
        in_sizes[0] != ag::MSH * ag::KD || in_sizes[1] != ag::KD * ag::NSH || out_size != ag::NR * ag::MSH * ag::NSH) {
        fprintf(stderr, "kernel_launch: unexpected problem shape\n");
        return;
    }
    {
        static bool attr_set[64];
        int dev = 0;
        HIP_CHECK(hipGetDevice(&dev));
        if ((unsigned)dev >= 64u || !attr_set[dev]) {
            HIP_CHECK(hipFuncSetAttribute((const void*)ag::ag_gemm_kernel, hipFuncAttributeMaxDynamicSharedMemorySize, (int)ag::LDS_BYTES));
            if ((unsigned)dev < 64u) attr_set[dev] = true;
        }
    }
    ag::Params p;
    p.x = (const uint8_t*)d_in[0];
    p.w = (const uint8_t*)d_in[1];
    p.sx = (const float*)d_in[2];
    p.sw = (const float*)d_in[3];
    p.out = (float*)d_out;
    for (int i = 0; i < ag::NR; ++i) p.ws[i] = (uint8_t*)d_ws_all[i];
    p.myws = (uint8_t*)d_ws_all[rank];
    p.rank = rank;
    hipLaunchKernelGGL(ag::ag_gemm_kernel, dim3(ag::NWG), dim3(ag::NTHR), ag::LDS_BYTES, stream, p);
    HIP_CHECK(hipGetLastError());
}
```
